```python
import numpy as np

kernel_name = "gpu_gemm_a2a_m4096_k4096_n16384_w8a8_dequant"


ABSMAX_RTOL = 0.001

COMPUTE_DTYPE = "int8"
EPILOGUE = "dequant"
IN_QUANT = "int8"
OUT_QUANT = None
IN_DTYPES = {"x": "int8", "w_mat": "int8",
             "scale_x": "float32", "scale_w": "float32"}
WORLD_SIZE = 8

def setup_inputs():

    x = np.random.randint(-127, 128, size=(4096, 4096)).astype(np.int8)
    w_mat = np.random.randint(-127, 128, size=(4096, 16384)).astype(np.int8)
    scale_x = (0.02 * (1.0 + np.random.rand(1))).astype(np.float32)
    scale_w = (0.02 * (1.0 + np.random.rand(1))).astype(np.float32)
    return {"x": x, "w_mat": w_mat, "scale_x": scale_x, "scale_w": scale_w}

def reference(x, w_mat, scale_x, scale_w):
    acc = x.astype(np.float64) @ w_mat.astype(np.float64)
    y = (acc.astype(np.float32) * (scale_x * scale_w)).astype(np.float32)
    return y
```

```cpp
#include <hip/hip_runtime.h>
#include <rccl/rccl.h>
#include <cstdio>
#include <cstdint>

#define HIP_CHECK(expr) do { hipError_t _e = (expr); if (_e != hipSuccess) { \
    fprintf(stderr, "HIP error %s at %s:%d\n", hipGetErrorString(_e), __FILE__, __LINE__); \
    return; } } while (0)

namespace {

constexpr int NR = 8;
constexpr int GM = 4096, GK = 4096, GN = 16384;
constexpr int MS = GM / NR;
constexpr int NS = GN / NR;
constexpr int BM = 128, BN = 128, BK = 128;
constexpr int NKT = GK / BK;
constexpr int NPH = 2;
constexpr int NSTAGE = 4;
constexpr int IMG = BM * BK;
constexpr int STAGE_BYTES = 2 * IMG;
constexpr int LDS_BYTES = NSTAGE * STAGE_BYTES;
constexpr int NWG = 256, NTHR = 512;

constexpr size_t WS_CTR = 0;
constexpr size_t WS_TMO = 2048;
constexpr size_t WS_FLAG = 4096;
constexpr size_t WS_PUB = 1u << 20;
constexpr size_t PUB_BYTES = (size_t)MS * GK;
constexpr size_t WS_NEED = WS_PUB + 2 * PUB_BYTES;

constexpr unsigned long long DEADLINE_TICKS = 50000000ull;

typedef int v2i __attribute__((ext_vector_type(2)));
typedef int v4i __attribute__((ext_vector_type(4)));
typedef float v4f __attribute__((ext_vector_type(4)));
typedef unsigned v4u __attribute__((ext_vector_type(4)));
#define LAS __attribute__((address_space(3)))

struct Params {
    const char* x;
    const char* w;
    const float* sx;
    const float* sw;
    float* out;
    char* ws[NR];
    int rank;
};

#define WAIT_V(n) asm volatile("s_waitcnt vmcnt(" #n ")" ::: "memory")
#define WAIT_L0() asm volatile("s_waitcnt lgkmcnt(0)" ::: "memory")
#define BAR() do { asm volatile("" ::: "memory"); __builtin_amdgcn_s_barrier(); asm volatile("" ::: "memory"); } while (0)

__global__ void __launch_bounds__(NTHR, 2) fused_gemm(Params p) {
    extern __shared__ __attribute__((aligned(16))) char shm[];
    LAS char* lds = (LAS char*)shm;
    const int tid = threadIdx.x, lane = tid & 63;
    const int wv = __builtin_amdgcn_readfirstlane(tid >> 6);
    const int b = blockIdx.x;
    char* me = p.ws[p.rank];

    unsigned* ctr = (unsigned*)(me + WS_CTR) + b;
    const int im = b >> 1;
    const int prow = (b & 1) * 64 + (tid >> 3);
    const int pslot = tid & 7;
    const char* xsrc = p.x + (size_t)((im >> 5) * BM + prow) * GK + (im & 31) * BK + ((pslot ^ ((prow >> 1) & 7)) << 4);
    const v4u xv = *(const v4u*)xsrc;
    float scale = p.sx[0] * p.sw[0];
    const unsigned call = __hip_atomic_load(ctr, __ATOMIC_RELAXED, __HIP_MEMORY_SCOPE_AGENT) + 1u;
    const size_t pub_off = WS_PUB + (size_t)(call & 1u) * PUB_BYTES;
    {
        __amdgpu_buffer_rsrc_t dst = __builtin_amdgcn_make_buffer_rsrc((void*)(me + pub_off + (size_t)b * 8192), 0, 8192, 0x00020000);
        __builtin_amdgcn_raw_buffer_store_b128(xv, dst, tid * 16, 0, 17  );
    }

    const int grp = b & 7, idx = b >> 3;
    const int src = (p.rank + grp) & (NR - 1);
    const int pl = idx >> 4, cn = idx & 15;
    const int wm = wv >> 2, wn = wv & 3;

    const char* wbase = p.w + (size_t)p.rank * NS + cn * BN;
    const unsigned bchunk = (unsigned)((lane & 7) ^ (((lane >> 4) & 3) | ((wv & 1) << 2)));
    const unsigned voffB = (unsigned)(wv * 16 + (lane >> 3)) * (unsigned)GN + (bchunk << 4);
    const unsigned voffA = (unsigned)(wv * 2048 + lane * 16);
    const char* abase = p.ws[src] + pub_off;

    auto stageB = [&](int st, int kt) {
        const char* g = wbase + (size_t)kt * ((size_t)BK * GN);
        LAS char* l = lds + st * STAGE_BYTES + IMG + wv * 2048;
        __builtin_amdgcn_global_load_lds((const unsigned*)(g + voffB), (LAS unsigned*)l, 16, 0, 0);
        __builtin_amdgcn_global_load_lds((const unsigned*)(g + voffB + 8u * GN), (LAS unsigned*)(l + 1024), 16, 0, 0);
    };
    auto stageA = [&](int st, int gstep) {
        const int panel = ((gstep >> 5) << 1) + pl;
        const char* g = abase + (size_t)(panel * NKT + (gstep & 31)) * IMG;
        LAS char* l = lds + st * STAGE_BYTES + wv * 2048;
        __builtin_amdgcn_global_load_lds((const unsigned*)(g + voffA), (LAS unsigned*)l, 16, 0, 0);
        __builtin_amdgcn_global_load_lds((const unsigned*)(g + voffA + 1024u), (LAS unsigned*)(l + 1024), 16, 0, 0);
    };

    asm volatile("" : "+v"(scale));
    WAIT_V(0);
    BAR();
    if (wv == 0) {
        if (lane < NR) {
            unsigned* f = (unsigned*)(p.ws[lane] + WS_FLAG) + p.rank * NWG + b;
            __hip_atomic_store(f, call, __ATOMIC_RELAXED, __HIP_MEMORY_SCOPE_SYSTEM);
        }
        if (lane == 0) __hip_atomic_store(ctr, call, __ATOMIC_RELAXED, __HIP_MEMORY_SCOPE_AGENT);
    }
    stageB(0, 0); stageB(1, 1); stageB(2, 2);

    unsigned* tmo = (unsigned*)(me + WS_TMO);
    LAS unsigned* lbad = (LAS unsigned*)(lds + 3 * STAGE_BYTES);
    if (wv == 0) {
        const unsigned* f0 = (const unsigned*)(me + WS_FLAG) + src * NWG + pl * 64 + lane;
        const unsigned* f1 = f0 + 128;
        const unsigned long long t0 = __builtin_amdgcn_s_memrealtime();
        unsigned bad = 0;
        for (;;) {
            const unsigned a = __hip_atomic_load(f0, __ATOMIC_RELAXED, __HIP_MEMORY_SCOPE_SYSTEM);
            const unsigned c = __hip_atomic_load(f1, __ATOMIC_RELAXED, __HIP_MEMORY_SCOPE_SYSTEM);
            if (__all((int)(a - call) >= 0 && (int)(c - call) >= 0)) break;
            if (__builtin_amdgcn_readfirstlane(__hip_atomic_load(tmo, __ATOMIC_RELAXED, __HIP_MEMORY_SCOPE_AGENT)) == call ||
                __builtin_amdgcn_s_memrealtime() - t0 > DEADLINE_TICKS) {
                if (lane == 0) __hip_atomic_store(tmo, call, __ATOMIC_RELAXED, __HIP_MEMORY_SCOPE_AGENT);
                bad = 1;
                break;
            }
            __builtin_amdgcn_s_sleep(1);
        }
        __builtin_amdgcn_fence(__ATOMIC_ACQUIRE, "");
        if (__builtin_amdgcn_readfirstlane(__hip_atomic_load(tmo, __ATOMIC_RELAXED, __HIP_MEMORY_SCOPE_AGENT)) == call) bad = 1;
        if (lane == 0) *lbad = bad;
        asm volatile("s_waitcnt vmcnt(0) lgkmcnt(0)" ::: "memory");
    }
    BAR();
    const bool bad = __builtin_amdgcn_readfirstlane(*lbad) != 0;
    WAIT_L0();
    BAR();

    float* obase = p.out + (size_t)(src * MS + pl * BM + wm * 64 + (lane & 15)) * NS + cn * BN + wn * 32 + (lane >> 4) * 4;

    if (bad) {
        const float q = __builtin_nanf("");
        const v4f qv = {q, q, q, q};
        for (int ph = 0; ph < NPH; ++ph)
            for (int mi = 0; mi < 4; ++mi)
                for (int ni = 0; ni < 2; ++ni)
                    *(v4f*)(obase + (size_t)(ph * 256 + mi * 16) * NS + ni * 16) = qv;
        WAIT_V(0);
        return;
    }

    stageA(0, 0); stageA(1, 1); stageA(2, 2);

    const int frow = lane & 15, fg = lane >> 4;
    const unsigned xa0 = (unsigned)((wm * 64 + frow) * 128 + ((fg ^ ((lane >> 1) & 7)) << 4));
    const unsigned xa1 = xa0 ^ 64u;
    const unsigned q = (unsigned)(frow >> 1), pb = (unsigned)(lane & 1);
    const unsigned fB = ((q >> 1) & 3u) | ((unsigned)(fg & 1) << 2);
    const unsigned wb0 = (unsigned)IMG + (unsigned)(16 * fg + q) * 128u + ((((unsigned)(wn * 2)) ^ fB) << 4) + 8u * pb;
    const unsigned wb1 = wb0 ^ 16u;

    v4i acc[4][2];
#pragma unroll
    for (int mi = 0; mi < 4; ++mi)
#pragma unroll
        for (int ni = 0; ni < 2; ++ni) acc[mi][ni] = (v4i){0, 0, 0, 0};

    auto compute = [&](int st) {
        const unsigned so = (unsigned)(st * STAGE_BYTES);
        const unsigned ax0 = xa0 + so, ax1 = xa1 + so, aw0 = wb0 + so, aw1 = wb1 + so;
        v2i wl[2][2], wh[2][2];
        v4i xf[2][4];
        asm volatile(
            "ds_read_b64_tr_b8 %0, %8\n\t"
            "ds_read_b64_tr_b8 %1, %8 offset:1024\n\t"
            "ds_read_b64_tr_b8 %2, %9\n\t"
            "ds_read_b64_tr_b8 %3, %9 offset:1024\n\t"
            "ds_read_b128 %4, %10\n\t"
            "ds_read_b128 %5, %10 offset:2048\n\t"
            "ds_read_b128 %6, %10 offset:4096\n\t"
            "ds_read_b128 %7, %10 offset:6144"
            : "=&v"(wl[0][0]), "=&v"(wh[0][0]), "=&v"(wl[0][1]), "=&v"(wh[0][1]),
              "=&v"(xf[0][0]), "=&v"(xf[0][1]), "=&v"(xf[0][2]), "=&v"(xf[0][3])
            : "v"(aw0), "v"(aw1), "v"(ax0) : "memory");
        asm volatile(
            "ds_read_b64_tr_b8 %0, %8 offset:8192\n\t"
            "ds_read_b64_tr_b8 %1, %8 offset:9216\n\t"
            "ds_read_b64_tr_b8 %2, %9 offset:8192\n\t"
            "ds_read_b64_tr_b8 %3, %9 offset:9216\n\t"
            "ds_read_b128 %4, %10\n\t"
            "ds_read_b128 %5, %10 offset:2048\n\t"
            "ds_read_b128 %6, %10 offset:4096\n\t"
            "ds_read_b128 %7, %10 offset:6144"
            : "=&v"(wl[1][0]), "=&v"(wh[1][0]), "=&v"(wl[1][1]), "=&v"(wh[1][1]),
              "=&v"(xf[1][0]), "=&v"(xf[1][1]), "=&v"(xf[1][2]), "=&v"(xf[1][3])
            : "v"(aw0), "v"(aw1), "v"(ax1) : "memory");
        asm volatile("s_waitcnt lgkmcnt(8)"
            : "+v"(wl[0][0]), "+v"(wh[0][0]), "+v"(wl[0][1]), "+v"(wh[0][1]),
              "+v"(xf[0][0]), "+v"(xf[0][1]), "+v"(xf[0][2]), "+v"(xf[0][3]) :: "memory");
        {
            const v4i w0 = {wl[0][0].x, wl[0][0].y, wh[0][0].x, wh[0][0].y};
            const v4i w1 = {wl[0][1].x, wl[0][1].y, wh[0][1].x, wh[0][1].y};
#pragma unroll
            for (int mi = 0; mi < 4; ++mi) {
                acc[mi][0] = __builtin_amdgcn_mfma_i32_16x16x64_i8(w0, xf[0][mi], acc[mi][0], 0, 0, 0);
                acc[mi][1] = __builtin_amdgcn_mfma_i32_16x16x64_i8(w1, xf[0][mi], acc[mi][1], 0, 0, 0);
            }
        }
        asm volatile("s_waitcnt lgkmcnt(0)"
            : "+v"(wl[1][0]), "+v"(wh[1][0]), "+v"(wl[1][1]), "+v"(wh[1][1]),
              "+v"(xf[1][0]), "+v"(xf[1][1]), "+v"(xf[1][2]), "+v"(xf[1][3]) :: "memory");
        {
            const v4i w0 = {wl[1][0].x, wl[1][0].y, wh[1][0].x, wh[1][0].y};
            const v4i w1 = {wl[1][1].x, wl[1][1].y, wh[1][1].x, wh[1][1].y};
#pragma unroll
            for (int mi = 0; mi < 4; ++mi) {
                acc[mi][0] = __builtin_amdgcn_mfma_i32_16x16x64_i8(w0, xf[1][mi], acc[mi][0], 0, 0, 0);
                acc[mi][1] = __builtin_amdgcn_mfma_i32_16x16x64_i8(w1, xf[1][mi], acc[mi][1], 0, 0, 0);
            }
        }
    };
    auto epilogue = [&](int ph) {
#pragma unroll
        for (int mi = 0; mi < 4; ++mi)
#pragma unroll
            for (int ni = 0; ni < 2; ++ni) {
                const v4i a = acc[mi][ni];
                v4f o = {(float)a.x * scale, (float)a.y * scale, (float)a.z * scale, (float)a.w * scale};
                __builtin_nontemporal_store(o, (v4f*)(obase + (size_t)(ph * 256 + mi * 16) * NS + ni * 16));
                acc[mi][ni] = (v4i){0, 0, 0, 0};
            }
    };

#define STEP(g, VM, PREFETCH) do { \
        WAIT_V(VM); WAIT_L0(); BAR(); \
        if (PREFETCH) { stageA(((g) + 3) & 3, (g) + 3); stageB(((g) + 3) & 3, ((g) + 3) & 31); } \
        compute((g) & 3); \
    } while (0)

    STEP(0, 4, true);
    STEP(1, 6, true);
    STEP(2, 8, true);
    STEP(3, 8, true);
    for (int g0 = 4; g0 < 28; g0 += 4) {
        STEP(g0 + 0, 8, true); STEP(g0 + 1, 8, true); STEP(g0 + 2, 8, true); STEP(g0 + 3, 8, true);
    }
    STEP(28, 8, true); STEP(29, 8, true); STEP(30, 8, true); STEP(31, 8, true);
    epilogue(0);
    STEP(32, 16, true);
    STEP(33, 16, true);
    STEP(34, 16, true);
    STEP(35, 8, true);
    for (int g0 = 36; g0 < 60; g0 += 4) {
        STEP(g0 + 0, 8, true); STEP(g0 + 1, 8, true); STEP(g0 + 2, 8, true); STEP(g0 + 3, 8, true);
    }
    STEP(60, 8, true);
    STEP(61, 8, false);
    STEP(62, 4, false);
    STEP(63, 0, false);
    epilogue(1);
    WAIT_V(0);
}

}

extern "C" void kernel_launch(int rank, int nranks, ncclComm_t comm,
                              hipStream_t stream,
                              void* const* d_in, const int* in_sizes, int n_in,
                              void* d_out, int out_size,
                              const int* in_dtypes, int out_dtype,
                              void* const* d_out_all,
                              void* const* d_ws_all, size_t ws_bytes) {
    (void)comm; (void)in_dtypes; (void)out_dtype; (void)d_out_all;
    if (nranks != NR || n_in != 4 || ws_bytes < WS_NEED || out_size != GM * NS ||
        in_sizes[0] != MS * GK || in_sizes[1] != GK * GN || (unsigned)rank >= (unsigned)NR) {
        fprintf(stderr, "kernel_launch: unexpected problem shape\n");
        return;
    }
    static bool attr_set = false;
    if (!attr_set) {
        HIP_CHECK(hipFuncSetAttribute((const void*)fused_gemm, hipFuncAttributeMaxDynamicSharedMemorySize, LDS_BYTES));
        attr_set = true;
    }
    Params p;
    p.x = (const char*)d_in[0];
    p.w = (const char*)d_in[1];
    p.sx = (const float*)d_in[2];
    p.sw = (const float*)d_in[3];
    p.out = (float*)d_out;
    for (int r = 0; r < NR; ++r) p.ws[r] = (char*)d_ws_all[r];
    p.rank = rank;
    fused_gemm<<<NWG, NTHR, LDS_BYTES, stream>>>(p);
    HIP_CHECK(hipGetLastError());
}
```

```cpp
#include <hip/hip_runtime.h>
#include <rccl/rccl.h>
#include <cstdio>
#include <cstdint>
#include <type_traits>

#define HIP_CHECK(expr) do { hipError_t _e = (expr); if (_e != hipSuccess) { \
    fprintf(stderr, "HIP error %s at %s:%d\n", hipGetErrorString(_e), __FILE__, __LINE__); \
    return; } } while (0)

namespace {

constexpr int NR = 8;
constexpr int GM = 4096, GK = 4096, GN = 16384;
constexpr int MS = GM / NR;
constexpr int NS = GN / NR;
constexpr int BM = 128, BN = 256, BK = 128;
constexpr int NKT = GK / BK;
constexpr int IMG = BM * BK;
constexpr int NSA = 4, NSB = 3;
constexpr int LDS_B = NSA * IMG;
constexpr int LDS_BYTES = NSA * IMG + NSB * 2 * IMG;
constexpr int NWG = 256, NTHR = 512;

constexpr size_t WS_CTR = 0;
constexpr size_t WS_TMO = 2048;
constexpr size_t WS_INV = 2304;
constexpr size_t WS_FLAG = 4096;
constexpr size_t WS_PUB = 1u << 20;
constexpr size_t PUB_BYTES = (size_t)MS * GK;
constexpr size_t WS_NEED = WS_PUB + 2 * PUB_BYTES;

constexpr unsigned long long DEADLINE_TICKS = 50000000ull;
constexpr unsigned long long INV_FALLBACK_TICKS = 300;

typedef int v2i __attribute__((ext_vector_type(2)));
typedef int v4i __attribute__((ext_vector_type(4)));
typedef float v4f __attribute__((ext_vector_type(4)));
typedef unsigned v4u __attribute__((ext_vector_type(4)));
#define LAS __attribute__((address_space(3)))
#define GAS __attribute__((address_space(1)))

struct Params {
    const char* x;
    const char* w;
    const float* sx;
    const float* sw;
    float* out;
    char* ws[NR];
    int rank;
};

#define GROT 0
#define OAUX 18
#define LOCAL_SLEEP 0
#define STAMP(i) do { } while (0)
#define WAIT_V(n) asm volatile("s_waitcnt vmcnt(" #n ")" ::: "memory")
#define WAIT_L0() asm volatile("s_waitcnt lgkmcnt(0)" ::: "memory")
#define BAR() do { asm volatile("" ::: "memory"); __builtin_amdgcn_s_barrier(); asm volatile("" ::: "memory"); } while (0)

struct Frag { v2i wl[2][4], wh[2][4]; v4i xf[2][4]; };

__global__ void __launch_bounds__(NTHR, 2) fused_gemm(Params p) {
    extern __shared__ __attribute__((aligned(16))) char shm[];
    LAS char* lds = (LAS char*)shm;
    const int tid = threadIdx.x, lane = tid & 63;
    const int wv = __builtin_amdgcn_readfirstlane(tid >> 6);
    const int b = blockIdx.x;
    char* me = p.ws[p.rank];
    STAMP(0);
    if (b < 8 && wv == 1) asm volatile("buffer_inv sc0 sc1" ::: "memory");
    const unsigned xcc = (unsigned)__builtin_amdgcn_s_getreg((3 << 11) | 20) & 0xFu;

    unsigned* ctr = (unsigned*)(me + WS_CTR) + b;
    const int im = b >> 1;
    const int prow = (b & 1) * 64 + (tid >> 3);
    const int pslot = tid & 7;
    const char* xsrc = p.x + (size_t)((im >> 5) * BM + prow) * GK + (im & 31) * BK + ((pslot ^ ((prow >> 1) & 7)) << 4);
    const v4u xv = *(const v4u*)xsrc;
    const float scale0 = p.sx[0] * p.sw[0];
    const unsigned call = __hip_atomic_load(ctr, __ATOMIC_RELAXED, __HIP_MEMORY_SCOPE_AGENT) + 1u;
    unsigned long long myflag = (unsigned long long)((unsigned*)(p.ws[lane & (NR - 1)] + WS_FLAG) + p.rank * NWG + b);
    const size_t pub_off = WS_PUB + (size_t)(call & 1u) * PUB_BYTES;
    {
        __amdgpu_buffer_rsrc_t dst = __builtin_amdgcn_make_buffer_rsrc((void*)(me + pub_off + (size_t)b * 8192), 0, 8192, 0x00020000);
        __builtin_amdgcn_raw_buffer_store_b128(xv, dst, tid * 16, 0, 17  );
    }

    const int grp = b & 7, idx = b >> 3;
    const int goff = (grp + GROT) & 7;
    const int src = (p.rank + goff) & (NR - 1);
    const int pl = idx >> 3, cn = idx & 7;
    const int wm = wv >> 2, wn = wv & 3;

    const unsigned bchunk = (unsigned)((lane & 7) ^ (((lane >> 4) & 3) | ((wv & 1) << 2)));
    const unsigned voffB = (unsigned)(wv * 16 + (lane >> 3)) * (unsigned)GN + (bchunk << 4);
    const unsigned voffA = (unsigned)(wv * 2048 + lane * 16);
    const __amdgpu_buffer_rsrc_t rsB = __builtin_amdgcn_make_buffer_rsrc((void*)(p.w + (size_t)p.rank * NS + cn * BN), 0,
                                                                         (int)((size_t)GK * GN - (size_t)p.rank * NS - cn * BN), 0x00020000);
    const __amdgpu_buffer_rsrc_t rsA = __builtin_amdgcn_make_buffer_rsrc((void*)(p.ws[src] + pub_off + (size_t)pl * NKT * IMG), 0,
                                                                         NKT * IMG, 0x00020000);
    auto stageB = [&](int st, int kt) {
        const int so = kt * (BK * GN);
        LAS char* l = lds + LDS_B + st * (2 * IMG) + wv * 2048;
        __builtin_amdgcn_raw_ptr_buffer_load_lds(rsB, (LAS void*)l, 16, voffB, so, 0, 0);
        __builtin_amdgcn_raw_ptr_buffer_load_lds(rsB, (LAS void*)(l + 1024), 16, voffB, so + 8 * GN, 0, 0);
        __builtin_amdgcn_raw_ptr_buffer_load_lds(rsB, (LAS void*)(l + IMG), 16, voffB, so + 128, 0, 0);
        __builtin_amdgcn_raw_ptr_buffer_load_lds(rsB, (LAS void*)(l + IMG + 1024), 16, voffB, so + 8 * GN + 128, 0, 0);
    };
    auto stageA = [&](int st, int kt) {
        LAS char* l = lds + st * IMG + wv * 2048;
        __builtin_amdgcn_raw_ptr_buffer_load_lds(rsA, (LAS void*)l, 16, voffA, kt * IMG, 0, 0);
        __builtin_amdgcn_raw_ptr_buffer_load_lds(rsA, (LAS void*)(l + 1024), 16, voffA, kt * IMG + 1024, 0, 0);
    };

    asm volatile("" : "+v"(myflag));
    const float scale = __builtin_bit_cast(float, __builtin_amdgcn_readfirstlane(__builtin_bit_cast(int, scale0)));
    WAIT_V(0);
    BAR();
    STAMP(1);
    unsigned* tmo = (unsigned*)(me + WS_TMO);
    unsigned* invw = (unsigned*)(me + WS_INV) + xcc * 4;
    LAS unsigned* lbad = (LAS unsigned*)(lds + 3 * IMG);
    if (wv == 0) {
        if (lane < NR) __hip_atomic_store((GAS unsigned*)myflag, call, __ATOMIC_RELAXED, __HIP_MEMORY_SCOPE_SYSTEM);
        if (lane == 8) __hip_atomic_store(ctr, call, __ATOMIC_RELAXED, __HIP_MEMORY_SCOPE_AGENT);
        const unsigned* f0 = (const unsigned*)(me + WS_FLAG) + src * NWG + pl * 64 + lane;
        const unsigned long long t0 = __builtin_amdgcn_s_memrealtime();
        unsigned bad = 0;
        for (;;) {
            const unsigned a = __hip_atomic_load(f0, __ATOMIC_RELAXED, __HIP_MEMORY_SCOPE_SYSTEM);
            const unsigned iv = __hip_atomic_load(invw, __ATOMIC_RELAXED, __HIP_MEMORY_SCOPE_AGENT);
            const unsigned tm = __hip_atomic_load(tmo, __ATOMIC_RELAXED, __HIP_MEMORY_SCOPE_AGENT);
            if (tm == call) { bad = 1; break; }
            const unsigned long long dt = __builtin_amdgcn_s_memrealtime() - t0;
            if (__all((int)(a - call) >= 0)) {
                if (iv == call) break;
                if (dt > INV_FALLBACK_TICKS) {
                    __builtin_amdgcn_fence(__ATOMIC_ACQUIRE, "");
                    break;
                }
            }
            if (dt > DEADLINE_TICKS) {
                if (lane == 0) __hip_atomic_store(tmo, call, __ATOMIC_RELAXED, __HIP_MEMORY_SCOPE_AGENT);
                bad = 1;
                break;
            }
            __builtin_amdgcn_s_sleep(1);
        }
        if (lane == 0) *lbad = bad;
        asm volatile("s_waitcnt vmcnt(0) lgkmcnt(0)" ::: "memory");
    } else if (wv == 1 && b < 8 && lane == 0) {
        __hip_atomic_store(invw, call, __ATOMIC_RELAXED, __HIP_MEMORY_SCOPE_AGENT);
    }
    stageB(0, 0); stageB(1, 1);
    BAR();
    const bool bad = __builtin_amdgcn_readfirstlane(*lbad) != 0;
    WAIT_L0();
    STAMP(2);

    if (bad) {
        float* obase = p.out + (size_t)(src * MS + pl * BM + wm * 64 + (lane & 15)) * NS + cn * BN + wn * 64 + (lane >> 4) * 4;
        const float q = __builtin_nanf("");
        const v4f qv = {q, q, q, q};
        for (int mi = 0; mi < 4; ++mi)
            for (int ni = 0; ni < 4; ++ni)
                *(v4f*)(obase + (size_t)(mi * 16) * NS + ni * 16) = qv;
        WAIT_V(0);
        return;
    }

    stageA(0, 0); stageA(1, 1); stageA(2, 2);

    const int frow = lane & 15, fg = lane >> 4;
    const unsigned xa0 = (unsigned)((wm * 64 + frow) * 128 + ((fg ^ ((lane >> 1) & 7)) << 4));
    const unsigned xa1 = xa0 ^ 64u;
    const unsigned q = (unsigned)(frow >> 1), pb = (unsigned)(lane & 1);
    const unsigned fB = ((q >> 1) & 3u) | ((unsigned)(fg & 1) << 2);
    const unsigned aw0 = (unsigned)(LDS_B + (wn >> 1) * IMG) + (unsigned)(16 * fg + q) * 128u + ((((unsigned)((wn & 1) * 4)) ^ fB) << 4) + 8u * pb;
    const unsigned aw1 = aw0 ^ 16u, aw2 = aw0 ^ 32u, aw3 = aw0 ^ 48u;

    v4i acc[4][4];
#pragma unroll
    for (int mi = 0; mi < 4; ++mi)
#pragma unroll
        for (int ni = 0; ni < 4; ++ni) acc[mi][ni] = (v4i){0, 0, 0, 0};
    Frag fr0, fr1;

    auto readfrag = [&](Frag& f, auto SA, auto SB) {
        constexpr int oa = decltype(SA)::value * IMG;
        constexpr int ob = (decltype(SB)::value & 1) * 2 * IMG;
        constexpr unsigned hb = (decltype(SB)::value >> 1) * 4u * IMG;
        const unsigned b0 = aw0 + hb, b1 = aw1 + hb, b2 = aw2 + hb, b3 = aw3 + hb;
        const unsigned x0 = xa0 + 0u, x1 = xa1 + 0u;
        asm volatile(
            "ds_read_b64_tr_b8 %0, %16 offset:%c20\n\t"
            "ds_read_b64_tr_b8 %1, %16 offset:%c20+1024\n\t"
            "ds_read_b64_tr_b8 %2, %17 offset:%c20\n\t"
            "ds_read_b64_tr_b8 %3, %17 offset:%c20+1024\n\t"
            "ds_read_b64_tr_b8 %4, %18 offset:%c20\n\t"
            "ds_read_b64_tr_b8 %5, %18 offset:%c20+1024\n\t"
            "ds_read_b64_tr_b8 %6, %19 offset:%c20\n\t"
            "ds_read_b64_tr_b8 %7, %19 offset:%c20+1024\n\t"
            "ds_read_b64_tr_b8 %8, %16 offset:%c20+8192\n\t"
            "ds_read_b64_tr_b8 %9, %16 offset:%c20+9216\n\t"
            "ds_read_b64_tr_b8 %10, %17 offset:%c20+8192\n\t"
            "ds_read_b64_tr_b8 %11, %17 offset:%c20+9216\n\t"
            "ds_read_b64_tr_b8 %12, %18 offset:%c20+8192\n\t"
            "ds_read_b64_tr_b8 %13, %18 offset:%c20+9216\n\t"
            "ds_read_b64_tr_b8 %14, %19 offset:%c20+8192\n\t"
            "ds_read_b64_tr_b8 %15, %19 offset:%c20+9216"
            : "=&v"(f.wl[0][0]), "=&v"(f.wh[0][0]), "=&v"(f.wl[0][1]), "=&v"(f.wh[0][1]),
              "=&v"(f.wl[0][2]), "=&v"(f.wh[0][2]), "=&v"(f.wl[0][3]), "=&v"(f.wh[0][3]),
              "=&v"(f.wl[1][0]), "=&v"(f.wh[1][0]), "=&v"(f.wl[1][1]), "=&v"(f.wh[1][1]),
              "=&v"(f.wl[1][2]), "=&v"(f.wh[1][2]), "=&v"(f.wl[1][3]), "=&v"(f.wh[1][3])
            : "v"(b0), "v"(b1), "v"(b2), "v"(b3), "i"(ob) : "memory");
        asm volatile(
            "ds_read_b128 %0, %8 offset:%c10\n\t"
            "ds_read_b128 %1, %8 offset:%c10+2048\n\t"
            "ds_read_b128 %2, %8 offset:%c10+4096\n\t"
            "ds_read_b128 %3, %8 offset:%c10+6144\n\t"
            "ds_read_b128 %4, %9 offset:%c10\n\t"
            "ds_read_b128 %5, %9 offset:%c10+2048\n\t"
            "ds_read_b128 %6, %9 offset:%c10+4096\n\t"
            "ds_read_b128 %7, %9 offset:%c10+6144"
            : "=&v"(f.xf[0][0]), "=&v"(f.xf[0][1]), "=&v"(f.xf[0][2]), "=&v"(f.xf[0][3]),
              "=&v"(f.xf[1][0]), "=&v"(f.xf[1][1]), "=&v"(f.xf[1][2]), "=&v"(f.xf[1][3])
            : "v"(x0), "v"(x1), "i"(oa) : "memory");
    };
    auto waitfrag = [&](Frag& f) {
        asm volatile("s_waitcnt lgkmcnt(0)"
            : "+v"(f.wl[0][0]), "+v"(f.wh[0][0]), "+v"(f.wl[0][1]), "+v"(f.wh[0][1]),
              "+v"(f.wl[0][2]), "+v"(f.wh[0][2]), "+v"(f.wl[0][3]), "+v"(f.wh[0][3]),
              "+v"(f.wl[1][0]), "+v"(f.wh[1][0]), "+v"(f.wl[1][1]), "+v"(f.wh[1][1]),
              "+v"(f.wl[1][2]), "+v"(f.wh[1][2]), "+v"(f.wl[1][3]), "+v"(f.wh[1][3]),
              "+v"(f.xf[0][0]), "+v"(f.xf[0][1]), "+v"(f.xf[0][2]), "+v"(f.xf[0][3]),
              "+v"(f.xf[1][0]), "+v"(f.xf[1][1]), "+v"(f.xf[1][2]), "+v"(f.xf[1][3]) :: "memory");
    };
    auto mma = [&](const Frag& f) {
        __builtin_amdgcn_s_setprio(1);
#pragma unroll
        for (int ks = 0; ks < 2; ++ks) {
#pragma unroll
            for (int ni = 0; ni < 4; ++ni) {
                const v4i w = {f.wl[ks][ni].x, f.wl[ks][ni].y, f.wh[ks][ni].x, f.wh[ks][ni].y};
#pragma unroll
                for (int mi = 0; mi < 4; ++mi)
                    acc[mi][ni] = __builtin_amdgcn_mfma_i32_16x16x64_i8(w, f.xf[ks][mi], acc[mi][ni], 0, 0, 0);
            }
        }
        __builtin_amdgcn_s_setprio(0);
    };

#define IC(n) std::integral_constant<int, (n)>{}
#define STEP(g, VM, PB, PA, FN, FP, DOMMA) do { \
        WAIT_V(VM); BAR(); \
        if (PB) stageB(((g) + 2) % 3, (g) + 2); \
        if (PA) stageA(((g) + 3) % 4, (g) + 3); \
        readfrag(FN, IC((g) % 4), IC((g) % 3)); \
        if (DOMMA) mma(FP); \
        if (LOCAL_SLEEP && goff == 0) __builtin_amdgcn_s_sleep(LOCAL_SLEEP); \
        waitfrag(FN); \
    } while (0)
#define STEP2(g) do { STEP((g), 8, true, true, fr0, fr1, true); STEP((g) + 1, 8, true, true, fr1, fr0, true); } while (0)

    STEP(0, 4, true, true, fr0, fr1, false);
    STAMP(3);
    STEP(1, 8, true, true, fr1, fr0, true);
    STEP2(2); STEP2(4); STEP2(6); STEP2(8); STEP2(10); STEP2(12); STEP2(14); STEP2(16); STEP2(18); STEP2(20);
    STEP2(22); STEP2(24); STEP2(26);
    STAMP(4);
    STEP(28, 8, true, true, fr0, fr1, true);
    STEP(29, 8, true, false, fr1, fr0, true);
    STEP(30, 6, false, false, fr0, fr1, true);
    STAMP(5);
    STEP(31, 0, false, false, fr1, fr0, true);
    mma(fr1);
    STAMP(6);
    int tz = threadIdx.x; asm volatile("" : "+v"(tz));
    const __amdgpu_buffer_rsrc_t rsO = __builtin_amdgcn_make_buffer_rsrc((void*)(p.out + (size_t)(src * MS + pl * BM) * NS + cn * BN), 0,
                                                                         (BM - 1) * NS * 4 + BN * 4, 0x00020000);
    const unsigned voffO = (unsigned)(((wm * 64 + (tz & 15)) * NS + wn * 64 + ((tz & 63) >> 4) * 4) * 4);
#pragma unroll
    for (int mi = 0; mi < 4; ++mi)
#pragma unroll
        for (int ni = 0; ni < 4; ++ni) {
            const v4i a = acc[mi][ni];
            const v4f o = {(float)a.x * scale, (float)a.y * scale, (float)a.z * scale, (float)a.w * scale};
            __builtin_amdgcn_raw_buffer_store_b128(__builtin_bit_cast(v4u, o), rsO, voffO + (unsigned)(mi * 16 * NS * 4 + ni * 64), 0, OAUX);
        }
    WAIT_V(0);
    STAMP(7);
}

}

extern "C" void kernel_launch(int rank, int nranks, ncclComm_t comm,
                              hipStream_t stream,
                              void* const* d_in, const int* in_sizes, int n_in,
                              void* d_out, int out_size,
                              const int* in_dtypes, int out_dtype,
                              void* const* d_out_all,
                              void* const* d_ws_all, size_t ws_bytes) {
    (void)comm; (void)in_dtypes; (void)out_dtype; (void)d_out_all;
    if (nranks != NR || n_in != 4 || ws_bytes < WS_NEED || out_size != GM * NS ||
        in_sizes[0] != MS * GK || in_sizes[1] != GK * GN || (unsigned)rank >= (unsigned)NR) {
        fprintf(stderr, "kernel_launch: unexpected problem shape\n");
        return;
    }
    static bool attr_set = false;
    if (!attr_set) {
        HIP_CHECK(hipFuncSetAttribute((const void*)fused_gemm, hipFuncAttributeMaxDynamicSharedMemorySize, LDS_BYTES));
        attr_set = true;
    }
    Params p;
    p.x = (const char*)d_in[0];
    p.w = (const char*)d_in[1];
    p.sx = (const float*)d_in[2];
    p.sw = (const float*)d_in[3];
    p.out = (float*)d_out;
    for (int r = 0; r < NR; ++r) p.ws[r] = (char*)d_ws_all[r];
    p.rank = rank;
    fused_gemm<<<NWG, NTHR, LDS_BYTES, stream>>>(p);
    HIP_CHECK(hipGetLastError());
}
```

```cpp
#include <hip/hip_runtime.h>
#include <rccl/rccl.h>
#include <cstdio>
#include <cstdint>
#include <type_traits>

#define HIP_CHECK(expr) do { hipError_t _e = (expr); if (_e != hipSuccess) { \
    fprintf(stderr, "HIP error %s at %s:%d\n", hipGetErrorString(_e), __FILE__, __LINE__); \
    return; } } while (0)

namespace {

constexpr int NR = 8;
constexpr int GM = 4096, GK = 4096, GN = 16384;
constexpr int MS = GM / NR;
constexpr int NS = GN / NR;
constexpr int BM = 128, BN = 256, BK = 128;
constexpr int NKT = GK / BK;
constexpr int IMG = BM * BK;
constexpr int NSA = 4, NSB = 3;
constexpr int LDS_B = NSA * IMG;
constexpr int LDS_BYTES = NSA * IMG + NSB * 2 * IMG;
constexpr int NWG = 256, NTHR = 512;

constexpr size_t WS_CTR = 0;
constexpr size_t WS_TMO = 2048;
constexpr size_t WS_INV = 2304;
constexpr size_t WS_FLAG = 4096;
constexpr size_t WS_PUB = 1u << 20;
constexpr size_t PUB_BYTES = (size_t)MS * GK;
constexpr size_t WS_NEED = WS_PUB + 2 * PUB_BYTES;

constexpr unsigned long long DEADLINE_TICKS = 50000000ull;
constexpr unsigned long long INV_FALLBACK_TICKS = 400;

typedef int v2i __attribute__((ext_vector_type(2)));
typedef int v4i __attribute__((ext_vector_type(4)));
typedef float v4f __attribute__((ext_vector_type(4)));
typedef unsigned v4u __attribute__((ext_vector_type(4)));
#define LAS __attribute__((address_space(3)))
#define GAS __attribute__((address_space(1)))

struct Params {
    const char* x;
    const char* w;
    const float* sx;
    const float* sw;
    float* out;
    char* me;
    char* ws[NR];
    int rank;
};

constexpr int OAUX = 18;

#define STAMP(i) do { } while (0)
#define WAIT_V(n) asm volatile("s_waitcnt vmcnt(" #n ")" ::: "memory")
#define WAIT_L0() asm volatile("s_waitcnt lgkmcnt(0)" ::: "memory")
#define BAR() do { asm volatile("" ::: "memory"); __builtin_amdgcn_s_barrier(); asm volatile("" ::: "memory"); } while (0)

struct Frag { v2i wl[2][4], wh[2][4]; v4i xf[2][4]; };

__global__ void __launch_bounds__(NTHR, 2) fused_gemm(Params p) {
    extern __shared__ __attribute__((aligned(16))) char shm[];
    LAS char* lds = (LAS char*)shm;
    const int tid = threadIdx.x, lane = tid & 63;
    const int wv = __builtin_amdgcn_readfirstlane(tid >> 6);
    const int b = blockIdx.x;
    char* me = p.me;
    STAMP(0);
    unsigned* ctr = (unsigned*)(me + WS_CTR) + b;
    const int im = b >> 1;
    const int prow = (b & 1) * 64 + (tid >> 3);
    const int pslot = tid & 7;
    const char* xsrc = p.x + (size_t)((im >> 5) * BM + prow) * GK + (im & 31) * BK + ((pslot ^ ((prow >> 1) & 7)) << 4);
    const float scale0 = p.sx[0] * p.sw[0];
    v4u xv; unsigned cv;
    asm volatile("global_load_dwordx4 %0, %2, off\n\tglobal_load_dword %1, %3, off sc1" : "=&v"(xv), "=&v"(cv) : "v"(xsrc), "v"(ctr) : "memory");
    if (b < 8 && wv == 1) asm volatile("buffer_inv sc0 sc1" ::: "memory");
    const unsigned xcc = (unsigned)__builtin_amdgcn_s_getreg((3 << 11) | 20) & 0xFu;
    unsigned long long myflag = (unsigned long long)((unsigned*)(p.ws[lane & (NR - 1)] + WS_FLAG) + p.rank * NWG + b);
    asm volatile("s_waitcnt vmcnt(0)" : "+v"(xv), "+v"(cv), "+v"(myflag) :: "memory");
    const unsigned call = (unsigned)__builtin_amdgcn_readfirstlane(cv) + 1u;
    const size_t pub_off = WS_PUB + (size_t)(call & 1u) * PUB_BYTES;
    {
        __amdgpu_buffer_rsrc_t dst = __builtin_amdgcn_make_buffer_rsrc((void*)(me + pub_off + (size_t)b * 8192), 0, 8192, 0x00020000);
        __builtin_amdgcn_raw_buffer_store_b128(xv, dst, tid * 16, 0, 17  );
    }

    const int grp = b & 7, idx = b >> 3;
    const int src = (p.rank + grp) & (NR - 1);
    const int pl = idx >> 3, cn = idx & 7;
    const int wm = wv >> 2, wn = wv & 3;

    const unsigned bchunk = (unsigned)((lane & 7) ^ (((lane >> 4) & 3) | ((wv & 1) << 2)));
    const unsigned voffB = (unsigned)(wv * 16 + (lane >> 3)) * (unsigned)GN + (bchunk << 4);
    const unsigned voffA = (unsigned)(wv * 2048 + lane * 16);
    const __amdgpu_buffer_rsrc_t rsB = __builtin_amdgcn_make_buffer_rsrc((void*)(p.w + (size_t)p.rank * NS + cn * BN), 0,
                                                                         (int)((size_t)GK * GN - (size_t)p.rank * NS - cn * BN), 0x00020000);
    const __amdgpu_buffer_rsrc_t rsA = __builtin_amdgcn_make_buffer_rsrc((void*)(p.ws[src] + pub_off + (size_t)pl * NKT * IMG), 0,
                                                                         NKT * IMG, 0x00020000);
    auto stageB = [&](int st, int kt) {
        const int so = kt * (BK * GN);
        LAS char* l = lds + LDS_B + st * (2 * IMG) + wv * 2048;
        __builtin_amdgcn_raw_ptr_buffer_load_lds(rsB, (LAS void*)l, 16, voffB, so, 0, 0);
        __builtin_amdgcn_raw_ptr_buffer_load_lds(rsB, (LAS void*)(l + 1024), 16, voffB, so + 8 * GN, 0, 0);
        __builtin_amdgcn_raw_ptr_buffer_load_lds(rsB, (LAS void*)(l + IMG), 16, voffB, so + 128, 0, 0);
        __builtin_amdgcn_raw_ptr_buffer_load_lds(rsB, (LAS void*)(l + IMG + 1024), 16, voffB, so + 8 * GN + 128, 0, 0);
    };
    auto stageA = [&](int st, int kt) {
        LAS char* l = lds + st * IMG + wv * 2048;
        __builtin_amdgcn_raw_ptr_buffer_load_lds(rsA, (LAS void*)l, 16, voffA, kt * IMG, 0, 0);
        __builtin_amdgcn_raw_ptr_buffer_load_lds(rsA, (LAS void*)(l + 1024), 16, voffA, kt * IMG + 1024, 0, 0);
    };

    const float scale1 = scale0;
    WAIT_V(0);
    BAR();
    STAMP(1);
    unsigned* tmo = (unsigned*)(me + WS_TMO);
    unsigned* invw = (unsigned*)(me + WS_INV) + xcc * 4;
    LAS unsigned* lbad = (LAS unsigned*)(lds + 3 * IMG);
    unsigned bad = 0;
    if (wv == 0) {
        if (lane < NR) __hip_atomic_store((GAS unsigned*)myflag, call, __ATOMIC_RELAXED, __HIP_MEMORY_SCOPE_SYSTEM);
        if (lane == 8) __hip_atomic_store(ctr, call, __ATOMIC_RELAXED, __HIP_MEMORY_SCOPE_AGENT);
        const unsigned* f0 = (const unsigned*)(me + WS_FLAG) + src * NWG + pl * 64 + lane;
        const unsigned lbad_addr = (unsigned)(uintptr_t)lbad, vzero = 0u;
        const unsigned long long t0 = __builtin_amdgcn_s_memrealtime();
        for (;;) {
            unsigned a0, a1, a2, c0, c1, c2, tt, ok, cnt, siv;
#define POLL_LD(A, C) "global_load_dword %" #A ", %10, off sc0 sc1\n\tglobal_load_dword %" #C ", %11, off sc0 sc1\n\t"
#define POLL_CK(A, C) "s_waitcnt vmcnt(4)\n\t" \
                      "v_subrev_u32 %6, %12, %" #A "\n\t" \
                      "v_cmp_gt_i32 vcc, 0, %6\n\t" \
                      "v_readfirstlane_b32 %9, %" #C "\n\t" \
                      "s_cmp_eq_u64 vcc, 0\n\t" \
                      "s_cselect_b32 %7, 1, 0\n\t" \
                      "s_cmp_eq_u32 %9, %12\n\t" \
                      "s_cselect_b32 %7, %7, 0\n\t" \
                      "s_cmp_eq_u32 %7, 1\n\t" \
                      "s_cbranch_scc1 2f\n\t" POLL_LD(A, C)
            asm volatile(
                POLL_LD(0, 3) POLL_LD(1, 4) POLL_LD(2, 5)
                "s_mov_b32 %8, 16\n"
                "1:\n\t"
                POLL_CK(0, 3) POLL_CK(1, 4) POLL_CK(2, 5)
                "s_sub_u32 %8, %8, 1\n\t"
                "s_cmp_lg_u32 %8, 0\n\t"
                "s_cbranch_scc1 1b\n\t"
                "s_mov_b32 %7, 0\n\t"
                "s_branch 3f\n"
                "2:\n\t"
                "ds_write_b32 %13, %14\n\t"
                "s_waitcnt lgkmcnt(0)\n\t"
                "s_barrier\n"
                "3:\n\t"
                "s_waitcnt vmcnt(0)"
                : "=&v"(a0), "=&v"(a1), "=&v"(a2), "=&v"(c0), "=&v"(c1), "=&v"(c2), "=&v"(tt), "=&s"(ok), "=&s"(cnt), "=&s"(siv)
                : "v"(f0), "v"(invw), "s"(call), "v"(lbad_addr), "v"(vzero)
                : "memory", "vcc", "scc");
            if (ok) break;
            const unsigned a = __hip_atomic_load(f0, __ATOMIC_RELAXED, __HIP_MEMORY_SCOPE_SYSTEM);
            const unsigned iv = __hip_atomic_load(invw, __ATOMIC_RELAXED, __HIP_MEMORY_SCOPE_AGENT);
            const unsigned tm = __hip_atomic_load(tmo, __ATOMIC_RELAXED, __HIP_MEMORY_SCOPE_AGENT);
            const unsigned long long dt = __builtin_amdgcn_s_memrealtime() - t0;
            bool done = false;
            if (tm == call) { bad = 1; done = true; }
            else if (__all((int)(a - call) >= 0) && (iv == call || dt > INV_FALLBACK_TICKS)) {
                if (iv != call) __builtin_amdgcn_fence(__ATOMIC_ACQUIRE, "");
                done = true;
            } else if (dt > DEADLINE_TICKS) {
                if (lane == 0) __hip_atomic_store(tmo, call, __ATOMIC_RELAXED, __HIP_MEMORY_SCOPE_AGENT);
                bad = 1; done = true;
            }
            if (done) {
                if (lane == 0) *lbad = bad;
                asm volatile("s_waitcnt vmcnt(0) lgkmcnt(0)" ::: "memory");
                BAR();
                break;
            }
        }
        stageB(0, 0); stageB(1, 1);
    } else {
        if (wv == 1 && b < 8 && lane == 0) __hip_atomic_store(invw, call, __ATOMIC_RELAXED, __HIP_MEMORY_SCOPE_AGENT);
        stageB(0, 0); stageB(1, 1);
        BAR();
        bad = __builtin_amdgcn_readfirstlane(*lbad);
        WAIT_L0();
    }
    STAMP(2);

    if (bad) {
        float* obase = p.out + (size_t)(src * MS + pl * BM + wm * 64 + (lane & 15)) * NS + cn * BN + wn * 64 + (lane >> 4) * 4;
        const float q = __builtin_nanf("");
        const v4f qv = {q, q, q, q};
        for (int mi = 0; mi < 4; ++mi)
            for (int ni = 0; ni < 4; ++ni)
                *(v4f*)(obase + (size_t)(mi * 16) * NS + ni * 16) = qv;
        WAIT_V(0);
        return;
    }

    stageA(0, 0); stageA(1, 1); stageA(2, 2);

    const int frow = lane & 15, fg = lane >> 4;
    const unsigned xa0 = (unsigned)((wm * 64 + frow) * 128 + ((fg ^ ((lane >> 1) & 7)) << 4));
    const unsigned xa1 = xa0 ^ 64u;
    const unsigned q = (unsigned)(frow >> 1), pb = (unsigned)(lane & 1);
    const unsigned fB = ((q >> 1) & 3u) | ((unsigned)(fg & 1) << 2);
    const unsigned aw0 = (unsigned)(LDS_B + (wn >> 1) * IMG) + (unsigned)(16 * fg + q) * 128u + ((((unsigned)((wn & 1) * 4)) ^ fB) << 4) + 8u * pb;
    const unsigned aw1 = aw0 ^ 16u, aw2 = aw0 ^ 32u, aw3 = aw0 ^ 48u;

    v4i acc[4][4];
#pragma unroll
    for (int mi = 0; mi < 4; ++mi)
#pragma unroll
        for (int ni = 0; ni < 4; ++ni) acc[mi][ni] = (v4i){0, 0, 0, 0};
    Frag fr0, fr1;

    auto readfrag = [&](Frag& f, auto SA, auto SB) {
        constexpr int oa = decltype(SA)::value * IMG;
        constexpr int ob = (decltype(SB)::value & 1) * 2 * IMG;
        constexpr unsigned hb = (decltype(SB)::value >> 1) * 4u * IMG;
        const unsigned b0 = aw0 + hb, b1 = aw1 + hb, b2 = aw2 + hb, b3 = aw3 + hb;
        const unsigned x0 = xa0 + 0u, x1 = xa1 + 0u;
        asm volatile(
            "ds_read_b64_tr_b8 %0, %16 offset:%c20\n\t"
            "ds_read_b64_tr_b8 %1, %16 offset:%c20+1024\n\t"
            "ds_read_b64_tr_b8 %2, %17 offset:%c20\n\t"
            "ds_read_b64_tr_b8 %3, %17 offset:%c20+1024\n\t"
            "ds_read_b64_tr_b8 %4, %18 offset:%c20\n\t"
            "ds_read_b64_tr_b8 %5, %18 offset:%c20+1024\n\t"
            "ds_read_b64_tr_b8 %6, %19 offset:%c20\n\t"
            "ds_read_b64_tr_b8 %7, %19 offset:%c20+1024\n\t"
            "ds_read_b64_tr_b8 %8, %16 offset:%c20+8192\n\t"
            "ds_read_b64_tr_b8 %9, %16 offset:%c20+9216\n\t"
            "ds_read_b64_tr_b8 %10, %17 offset:%c20+8192\n\t"
            "ds_read_b64_tr_b8 %11, %17 offset:%c20+9216\n\t"
            "ds_read_b64_tr_b8 %12, %18 offset:%c20+8192\n\t"
            "ds_read_b64_tr_b8 %13, %18 offset:%c20+9216\n\t"
            "ds_read_b64_tr_b8 %14, %19 offset:%c20+8192\n\t"
            "ds_read_b64_tr_b8 %15, %19 offset:%c20+9216"
            : "=&v"(f.wl[0][0]), "=&v"(f.wh[0][0]), "=&v"(f.wl[0][1]), "=&v"(f.wh[0][1]),
              "=&v"(f.wl[0][2]), "=&v"(f.wh[0][2]), "=&v"(f.wl[0][3]), "=&v"(f.wh[0][3]),
              "=&v"(f.wl[1][0]), "=&v"(f.wh[1][0]), "=&v"(f.wl[1][1]), "=&v"(f.wh[1][1]),
              "=&v"(f.wl[1][2]), "=&v"(f.wh[1][2]), "=&v"(f.wl[1][3]), "=&v"(f.wh[1][3])
            : "v"(b0), "v"(b1), "v"(b2), "v"(b3), "i"(ob) : "memory");
        asm volatile(
            "ds_read_b128 %0, %8 offset:%c10\n\t"
            "ds_read_b128 %1, %8 offset:%c10+2048\n\t"
            "ds_read_b128 %2, %8 offset:%c10+4096\n\t"
            "ds_read_b128 %3, %8 offset:%c10+6144\n\t"
            "ds_read_b128 %4, %9 offset:%c10\n\t"
            "ds_read_b128 %5, %9 offset:%c10+2048\n\t"
            "ds_read_b128 %6, %9 offset:%c10+4096\n\t"
            "ds_read_b128 %7, %9 offset:%c10+6144"
            : "=&v"(f.xf[0][0]), "=&v"(f.xf[0][1]), "=&v"(f.xf[0][2]), "=&v"(f.xf[0][3]),
              "=&v"(f.xf[1][0]), "=&v"(f.xf[1][1]), "=&v"(f.xf[1][2]), "=&v"(f.xf[1][3])
            : "v"(x0), "v"(x1), "i"(oa) : "memory");
    };
    auto waitfrag = [&](Frag& f) {
        asm volatile("s_waitcnt lgkmcnt(0)"
            : "+v"(f.wl[0][0]), "+v"(f.wh[0][0]), "+v"(f.wl[0][1]), "+v"(f.wh[0][1]),
              "+v"(f.wl[0][2]), "+v"(f.wh[0][2]), "+v"(f.wl[0][3]), "+v"(f.wh[0][3]),
              "+v"(f.wl[1][0]), "+v"(f.wh[1][0]), "+v"(f.wl[1][1]), "+v"(f.wh[1][1]),
              "+v"(f.wl[1][2]), "+v"(f.wh[1][2]), "+v"(f.wl[1][3]), "+v"(f.wh[1][3]),
              "+v"(f.xf[0][0]), "+v"(f.xf[0][1]), "+v"(f.xf[0][2]), "+v"(f.xf[0][3]),
              "+v"(f.xf[1][0]), "+v"(f.xf[1][1]), "+v"(f.xf[1][2]), "+v"(f.xf[1][3]) :: "memory");
    };
    auto mma = [&](const Frag& f) {
        __builtin_amdgcn_s_setprio(1);
#pragma unroll
        for (int ks = 0; ks < 2; ++ks) {
#pragma unroll
            for (int ni = 0; ni < 4; ++ni) {
                const v4i w = {f.wl[ks][ni].x, f.wl[ks][ni].y, f.wh[ks][ni].x, f.wh[ks][ni].y};
#pragma unroll
                for (int mi = 0; mi < 4; ++mi)
                    acc[mi][ni] = __builtin_amdgcn_mfma_i32_16x16x64_i8(w, f.xf[ks][mi], acc[mi][ni], 0, 0, 0);
            }
        }
        __builtin_amdgcn_s_setprio(0);
    };

#define IC(n) std::integral_constant<int, (n)>{}
#define STEP(g, VM, PB, PA, FN, FP, DOMMA) do { \
        WAIT_V(VM); BAR(); \
        if (PB) stageB(((g) + 2) % 3, (g) + 2); \
        if (PA) stageA(((g) + 3) % 4, (g) + 3); \
        readfrag(FN, IC((g) % 4), IC((g) % 3)); \
        if (DOMMA) mma(FP); \
        waitfrag(FN); \
    } while (0)
#define STEP2(g) do { STEP((g), 8, true, true, fr0, fr1, true); STEP((g) + 1, 8, true, true, fr1, fr0, true); } while (0)

    STEP(0, 4, true, true, fr0, fr1, false);
    STAMP(3);
    STEP(1, 8, true, true, fr1, fr0, true);
    STEP2(2); STEP2(4); STEP2(6); STEP2(8); STEP2(10); STEP2(12); STEP2(14); STEP2(16); STEP2(18); STEP2(20);
    STEP2(22); STEP2(24); STEP2(26);
    STAMP(4);
    unsigned tmv;
    asm volatile("global_load_dword %0, %1, off sc1" : "=v"(tmv) : "v"(tmo) : "memory");
    STEP(28, 9, true, true, fr0, fr1, true);
    STEP(29, 8, true, false, fr1, fr0, true);
    STEP(30, 6, false, false, fr0, fr1, true);
    STAMP(5);
    mma(fr0);
    WAIT_V(0); BAR();
    readfrag(fr1, IC(31 % 4), IC(31 % 3));
    waitfrag(fr1);
#pragma unroll
    for (int mi = 0; mi < 4; ++mi)
#pragma unroll
        for (int ni = 0; ni < 4; ++ni)
#pragma unroll
            for (int ks = 0; ks < 2; ++ks) {
                const v4i w = {fr1.wl[ks][ni].x, fr1.wl[ks][ni].y, fr1.wh[ks][ni].x, fr1.wh[ks][ni].y};
                acc[mi][ni] = __builtin_amdgcn_mfma_i32_16x16x64_i8(w, fr1.xf[ks][mi], acc[mi][ni], 0, 0, 0);
            }
    STAMP(6);
    asm volatile("" : "+v"(tmv));
    const float scale = (__builtin_amdgcn_readfirstlane(tmv) == call) ? __builtin_nanf("") : scale1;
    int tz = threadIdx.x; asm volatile("" : "+v"(tz));
    const __amdgpu_buffer_rsrc_t rsO = __builtin_amdgcn_make_buffer_rsrc((void*)(p.out + (size_t)(src * MS + pl * BM) * NS + cn * BN), 0,
                                                                         (BM - 1) * NS * 4 + BN * 4, 0x00020000);
    const unsigned voffO = (unsigned)(((wm * 64 + (tz & 7)) * NS + wn * 64 + ((tz & 63) >> 4) * 4 + ((tz & 8) ? 16 : 0)) * 4);
#pragma unroll
    for (int mi = 0; mi < 4; ++mi)
#pragma unroll
        for (int np = 0; np < 2; ++np) {
            const v4i a = acc[mi][2 * np], c = acc[mi][2 * np + 1];
            v4i X, Y;
#pragma unroll
            for (int j = 0; j < 4; ++j) {
                X[j] = __builtin_amdgcn_update_dpp(a[j], c[j], 0x128  , 0xf, 0xc, false);
                Y[j] = __builtin_amdgcn_update_dpp(c[j], a[j], 0x128, 0xf, 0x3, false);
            }
            const v4f ox = {(float)X.x * scale, (float)X.y * scale, (float)X.z * scale, (float)X.w * scale};
            const v4f oy = {(float)Y.x * scale, (float)Y.y * scale, (float)Y.z * scale, (float)Y.w * scale};
            __builtin_amdgcn_raw_buffer_store_b128(__builtin_bit_cast(v4u, ox), rsO, voffO + (unsigned)(mi * 16 * NS * 4 + np * 128), 0, OAUX);
            __builtin_amdgcn_raw_buffer_store_b128(__builtin_bit_cast(v4u, oy), rsO, voffO + (unsigned)((mi * 16 + 8) * NS * 4 + np * 128), 0, OAUX);
        }
    WAIT_V(0);
    STAMP(7);
}

}

extern "C" void kernel_launch(int rank, int nranks, ncclComm_t comm,
                              hipStream_t stream,
                              void* const* d_in, const int* in_sizes, int n_in,
                              void* d_out, int out_size,
                              const int* in_dtypes, int out_dtype,
                              void* const* d_out_all,
                              void* const* d_ws_all, size_t ws_bytes) {
    (void)comm; (void)in_dtypes; (void)out_dtype; (void)d_out_all;
    if (nranks != NR || n_in != 4 || ws_bytes < WS_NEED || out_size != GM * NS ||
        in_sizes[0] != MS * GK || in_sizes[1] != GK * GN || (unsigned)rank >= (unsigned)NR) {
        fprintf(stderr, "kernel_launch: unexpected problem shape\n");
        return;
    }
    static bool attr_set = false;
    if (!attr_set) {
        HIP_CHECK(hipFuncSetAttribute((const void*)fused_gemm, hipFuncAttributeMaxDynamicSharedMemorySize, LDS_BYTES));
        attr_set = true;
    }
    Params p;
    p.x = (const char*)d_in[0];
    p.w = (const char*)d_in[1];
    p.sx = (const float*)d_in[2];
    p.sw = (const float*)d_in[3];
    p.out = (float*)d_out;
    for (int r = 0; r < NR; ++r) p.ws[r] = (char*)d_ws_all[r];
    p.rank = rank;
    p.me = (char*)d_ws_all[rank];
    fused_gemm<<<NWG, NTHR, LDS_BYTES, stream>>>(p);
    HIP_CHECK(hipGetLastError());
}
```
